```python
import numpy as np

kernel_name = "gpu_gemm_rs_m8192_k8192_n8192_f32_silu"


ABSMAX_RTOL = 0.001

COMPUTE_DTYPE = "float32"
EPILOGUE = "silu"
IN_QUANT = None
OUT_QUANT = None
IN_DTYPES = {"x": "float32", "w_mat": "float32"}
WORLD_SIZE = 8

def setup_inputs():

    x = np.random.randn(8192, 8192).astype(np.float32)
    w_mat = (np.random.randn(8192, 8192) / np.sqrt(8192)).astype(np.float32)
    return {"x": x, "w_mat": w_mat}

def _epilogue(y):
    return (y / (1.0 + np.exp(-np.clip(y, -60.0, 60.0)))).astype(np.float32)

def reference(x, w_mat):
    y = (x @ w_mat).astype(np.float32)
    return _epilogue(y)
```

```cpp
#include <hip/hip_runtime.h>
#include <rccl/rccl.h>
#include <cstdio>

#define HIP_CHECK(expr) do { hipError_t _e = (expr); if (_e != hipSuccess) { \
    fprintf(stderr, "HIP error %s at %s:%d\n", hipGetErrorString(_e), __FILE__, __LINE__); \
    return; } } while (0)

namespace gk {

#define LAS __attribute__((address_space(3)))
#define GAS __attribute__((address_space(1)))
typedef _Float16 h16x8 __attribute__((ext_vector_type(8)));
typedef _Float16 h16x2 __attribute__((ext_vector_type(2)));
typedef float f32x4 __attribute__((ext_vector_type(4)));
typedef unsigned u32x4 __attribute__((ext_vector_type(4)));
typedef GAS unsigned gu32;

constexpr int NR = 8;
constexpr int MG = 8192, NG = 8192;
constexpr int KS = 1024;
constexpr int NKT = 128;
constexpr int NRT = 8, NCT = 16;
constexpr int BLK = 32768;
constexpr int HTB = 16384;

constexpr size_t WS_DONE = 0;
constexpr size_t WS_TMO = 128;
constexpr size_t WS_FLX = 4096;
constexpr size_t WS_FLW = WS_FLX + NRT * NR * 128;
constexpr size_t WS_FLO = WS_FLW + NCT * NR * 128;
constexpr size_t WS_DBG = 65536;
constexpr size_t WS_X = 1u << 20;
constexpr size_t XBUF = (size_t)NRT * NKT * BLK;
constexpr size_t WS_W = WS_X + 2 * XBUF;
constexpr size_t WBUF = (size_t)NCT * NKT * BLK;
constexpr size_t WS_END = WS_W + 2 * WBUF;

constexpr unsigned long long DEADLINE = 60000000ull;

#define PUSH_PER_LINK 8
#define PUSH_SELF 8
constexpr int GL = PUSH_PER_LINK, GS = PUSH_SELF;
constexpr int NPUSH = 7 * GL + GS;
constexpr int NPUSH_PAD = (NPUSH + 7) / 8 * 8;
constexpr int NGEMM = NRT * NCT;
constexpr int GRID = NPUSH_PAD + NGEMM;
static_assert(GRID <= 256, "one workgroup per CU");
constexpr int UNITS = 16 * (NRT + NCT);
constexpr int LDS_MAIN = 8 * HTB;
constexpr int LDS_BYTES = LDS_MAIN + 2048;

struct Params {
    const float* x; const float* w;
    char* ws[NR]; float* out[NR];
    int rank; int pad;
};

__device__ __forceinline__ unsigned long long rtc() { return __builtin_amdgcn_s_memrealtime(); }
__device__ __forceinline__ unsigned ld_sys(const unsigned* p) { return __hip_atomic_load((const gu32*)p, __ATOMIC_RELAXED, __HIP_MEMORY_SCOPE_SYSTEM); }
__device__ __forceinline__ unsigned ld_agt(const unsigned* p) { return __hip_atomic_load((const gu32*)p, __ATOMIC_RELAXED, __HIP_MEMORY_SCOPE_AGENT); }
__device__ __forceinline__ void st_sys(unsigned* p, unsigned v) { __hip_atomic_store((gu32*)p, v, __ATOMIC_RELAXED, __HIP_MEMORY_SCOPE_SYSTEM); }
__device__ __forceinline__ void st_agt(unsigned* p, unsigned v) { __hip_atomic_store((gu32*)p, v, __ATOMIC_RELAXED, __HIP_MEMORY_SCOPE_AGENT); }
__device__ __forceinline__ void st16_wt(void* p, u32x4 v) { asm volatile("global_store_dwordx4 %0, %1, off sc0 sc1\n\ts_nop 1" :: "v"(p), "v"(v) : "memory"); }
__device__ __forceinline__ unsigned pk2(float a, float b) { h16x2 v; v.x = (_Float16)a; v.y = (_Float16)b; return __builtin_bit_cast(unsigned, v); }
__device__ __forceinline__ int uni(int v) { return __builtin_amdgcn_readfirstlane(v); }

#define STAMP(slot) do {} while (0)

__device__ __forceinline__ int row_base(int a, int i) { return i < 4 ? a * 1024 + i * 256 : ((((a >> 2) ^ 1) << 2) + (i - 4)) * 1024 + (a & 3) * 256; }

struct Unit { const char* src; char* dst; unsigned* flag; int isw; };

__device__ __forceinline__ void decode_unit(const Params& P, int dest, int n, int par, int wid, int lane, Unit& u) {
    const int s = n / 24, r = n - s * 24, kt = P.rank * 16 + s;
    char* dws = P.ws[dest];
    if (r >= 4 && r < 20) {
        const int j = r - 4;
        u.isw = 1;
        u.src = (const char*)P.w + ((size_t)(s * 64 + wid * 8) * NG + (size_t)((dest >> 2) * 4096 + j * 256 + lane * 4)) * 4;
        u.dst = dws + WS_W + par * WBUF + (size_t)(j * NKT + kt) * BLK + (lane >> 5) * HTB + wid * 2048;
        u.flag = (unsigned*)(dws + WS_FLW) + (j * NR + P.rank) * 32 + s;
    } else {
        const int i = r < 4 ? 4 + r : r - 20;
        const int row = wid * 32 + (lane >> 3);
        u.isw = 0;
        u.src = (const char*)P.x + ((size_t)(row_base(dest, i) + row) * KS + s * 64 + (lane & 7) * 8) * 4;
        u.dst = dws + WS_X + par * XBUF + (size_t)(i * NKT + kt) * BLK + (row >> 7) * HTB + (row & 127) * 128 + (((lane & 7) ^ (row & 7)) << 4);
        u.flag = (unsigned*)(dws + WS_FLX) + (i * NR + P.rank) * 32 + s;
    }
}
__device__ __forceinline__ void load_unit(const Unit& u, f32x4 (&v)[8]) {
    const size_t s1 = u.isw ? 32768 : 16, s2 = u.isw ? 65536 : 32768;
#pragma unroll
    for (int i = 0; i < 8; ++i) v[i] = __builtin_nontemporal_load((const f32x4*)(u.src + (i >> 1) * s2 + (i & 1) * s1));
}
__device__ __forceinline__ void store_unit(const Unit& u, const f32x4 (&v)[8], int lane) {
    if (u.isw) {
#pragma unroll
        for (int j = 0; j < 4; ++j) {
            u32x4 o; o.x = pk2(v[0][j], v[1][j]); o.y = pk2(v[2][j], v[3][j]); o.z = pk2(v[4][j], v[5][j]); o.w = pk2(v[6][j], v[7][j]);
            st16_wt(u.dst + (j * 32 + (((lane & 31) + 4 * j) & 31)) * 16, o);
        }
    } else {
#pragma unroll
        for (int t = 0; t < 4; ++t) {
            u32x4 o; o.x = pk2(v[2 * t][0], v[2 * t][1]); o.y = pk2(v[2 * t][2], v[2 * t][3]); o.z = pk2(v[2 * t + 1][0], v[2 * t + 1][1]); o.w = pk2(v[2 * t + 1][2], v[2 * t + 1][3]);
            st16_wt(u.dst + t * 1024, o);
        }
    }
}
__device__ __forceinline__ void publish_unit(LAS unsigned* cnt, int idx, unsigned* flag, unsigned call, int lane) {
    if (lane == 0) {
        const unsigned old = __hip_atomic_fetch_add(cnt + idx, 1u, __ATOMIC_RELAXED, __HIP_MEMORY_SCOPE_WORKGROUP);
        if (old == 7u) st_sys(flag, call);
    }
}

__device__ __forceinline__ void pusher(const Params& P, LAS unsigned char* lds, int dest, int v, int g, unsigned call) {
    const int tid = threadIdx.x, wid = uni(tid >> 6), lane = tid & 63, par = call & 1;
    LAS unsigned* cnt = (LAS unsigned*)lds;
    if (tid < 512) cnt[tid] = 0u;
    __syncthreads();
    const int n_units = (UNITS - v + g - 1) / g;
    Unit cur, nxt; f32x4 a[8], b[8];
    unsigned* prev_flag = nullptr;
    decode_unit(P, dest, v, par, wid, lane, cur);
    load_unit(cur, a);
    for (int k = 0; k < n_units; k += 2) {
        if (k + 1 < n_units) {
            decode_unit(P, dest, v + (k + 1) * g, par, wid, lane, nxt);
            load_unit(nxt, b);
            asm volatile("s_waitcnt vmcnt(8)" ::: "memory");
        } else {
            asm volatile("s_waitcnt vmcnt(0)" ::: "memory");
        }
        if (k > 0) publish_unit(cnt, k - 1, prev_flag, call, lane);
        store_unit(cur, a, lane);
        prev_flag = cur.flag;
        if (k + 1 >= n_units) { asm volatile("s_waitcnt vmcnt(0)" ::: "memory"); publish_unit(cnt, k, prev_flag, call, lane); break; }
        if (k + 2 < n_units) {
            decode_unit(P, dest, v + (k + 2) * g, par, wid, lane, cur);
            load_unit(cur, a);
            asm volatile("s_waitcnt vmcnt(8)" ::: "memory");
        } else {
            asm volatile("s_waitcnt vmcnt(0)" ::: "memory");
        }
        publish_unit(cnt, k, prev_flag, call, lane);
        store_unit(nxt, b, lane);
        prev_flag = nxt.flag;
        if (k + 2 >= n_units) { asm volatile("s_waitcnt vmcnt(0)" ::: "memory"); publish_unit(cnt, k + 1, prev_flag, call, lane); break; }
    }
}

#define LD_AUX 17
__device__ __forceinline__ float silu(float y) {
    const float c = __builtin_amdgcn_fmed3f(y, -60.0f, 60.0f);
    return y * __builtin_amdgcn_rcpf(1.0f + __expf(-c));
}

__device__ __forceinline__ void gemm_tile(const Params& P, LAS unsigned char* lds, int ti, int tj, unsigned call) {
    const int tid = threadIdx.x, wid = uni(tid >> 6), lane = tid & 63, wr = wid >> 2, wc = wid & 3, fr = lane & 15, fq = lane >> 4;
    char* ws = P.ws[P.rank];
    const char* xb = ws + WS_X + (call & 1) * XBUF + (size_t)ti * NKT * BLK;
    const char* wb = ws + WS_W + (call & 1) * WBUF + (size_t)tj * NKT * BLK;
    const unsigned* fX = (const unsigned*)(ws + WS_FLX) + ti * NR * 32;
    const unsigned* fW = (const unsigned*)(ws + WS_FLW) + tj * NR * 32;
    unsigned* tmo = (unsigned*)(ws + WS_TMO);
    LAS u32x4* mslot = (LAS u32x4*)(lds + LDS_MAIN);
    const unsigned voff = (unsigned)tid * 16u;
    const unsigned ldsw = (unsigned)wid * 1024u;
    const int aoff0 = (wr * 64 + fr) * 128 + ((fq ^ (fr & 7)) << 4), aoff1 = aoff0 ^ 64;
    const int boff0 = fq * 2048 + ((fr & 3) * 32 + ((wc * 8 + (fr >> 2) + 4 * (fr & 3)) & 31)) * 16;
    const int boff1 = fq * 2048 + ((fr & 3) * 32 + ((wc * 8 + 4 + (fr >> 2) + 4 * (fr & 3)) & 31)) * 16;
#define SA(b, h) (((b) * 2 + (h)) * HTB)
#define SB(b, h) ((4 + (b) * 2 + (h)) * HTB)
#define STAGE(bufoff, gbase) do { _Pragma("unroll") for (int _i = 0; _i < 2; ++_i) \
        __builtin_amdgcn_global_load_lds((const unsigned*)((gbase) + _i * 8192 + voff), (LAS unsigned*)(lds + (bufoff) + ldsw + _i * 8192), 16, 0, LD_AUX); } while (0)
#define LDA(dst, b, h) do { _Pragma("unroll") for (int m = 0; m < 4; ++m) { dst[m][0] = *(const LAS h16x8*)(lds + SA(b, h) + aoff0 + m * 2048); dst[m][1] = *(const LAS h16x8*)(lds + SA(b, h) + aoff1 + m * 2048); } } while (0)
#define LDB(dst, b, h) do { _Pragma("unroll") for (int k = 0; k < 2; ++k) { dst[0][k] = *(const LAS h16x8*)(lds + SB(b, h) + boff0 + k * 8192); dst[1][k] = *(const LAS h16x8*)(lds + SB(b, h) + boff1 + k * 8192); } } while (0)
#define MMA(ai, bj, At, Bt) do { __builtin_amdgcn_s_setprio(1); _Pragma("unroll") for (int m = 0; m < 4; ++m) _Pragma("unroll") for (int n = 0; n < 2; ++n) _Pragma("unroll") for (int k = 0; k < 2; ++k) \
        acc[ai][bj][m][n] = __builtin_amdgcn_mfma_f32_16x16x32_f16(Bt[n][k], At[m][k], acc[ai][bj][m][n], 0, 0, 0); __builtin_amdgcn_s_setprio(0); } while (0)
#define WAIT_V(n) asm volatile("s_waitcnt vmcnt(" #n ")" ::: "memory")
#define WAIT_L(n) asm volatile("s_waitcnt lgkmcnt(" #n ")" ::: "memory")
#define BAR __builtin_amdgcn_s_barrier()
#define SCHED __builtin_amdgcn_sched_barrier(0)

    f32x4 acc[2][2][4][2];
#pragma unroll
    for (int a = 0; a < 2; ++a)
#pragma unroll
        for (int b = 0; b < 2; ++b)
#pragma unroll
            for (int m = 0; m < 4; ++m)
#pragma unroll
                for (int n = 0; n < 2; ++n) acc[a][b][m][n] = (f32x4){0.f, 0.f, 0.f, 0.f};
    h16x8 At[4][2], B0[2][2], B1[2][2];

    unsigned long long avLo = 0, avHi = 0, isLo = 0, isHi = 0;
    int n_issued = 0, last_kt = 0;
    auto pick = [&](bool block, bool& valid) -> int {
        if ((avLo | avHi) == 0ull && n_issued < NKT) {
            if (wid == 0) {
                const int o = (lane >> 4) * 32 + (lane & 15);
                const unsigned long long t0 = rtc();
                unsigned long long lo, hi;
                for (;;) {
                    const unsigned x0 = ld_sys(fX + o), x1 = ld_sys(fX + 128 + o), w0 = ld_sys(fW + o), w1 = ld_sys(fW + 128 + o);
                    lo = __ballot(x0 >= call && w0 >= call) & ~isLo; hi = __ballot(x1 >= call && w1 >= call) & ~isHi;
                    if (!block || (lo | hi) != 0ull) break;
                    if ((unsigned)uni((int)ld_agt(tmo)) == call || rtc() - t0 > DEADLINE) { if (lane == 0) st_agt(tmo, call); lo = ~isLo; hi = ~isHi; break; }
                    __builtin_amdgcn_s_sleep(4);
                }
                if (lane == 0) *mslot = (u32x4){(unsigned)lo, (unsigned)(lo >> 32), (unsigned)hi, (unsigned)(hi >> 32)};
                asm volatile("s_waitcnt vmcnt(0) lgkmcnt(0)" ::: "memory");
            }
            asm volatile("" ::: "memory"); BAR; asm volatile("" ::: "memory");
            const u32x4 mv = *mslot;
            avLo = ((unsigned long long)(unsigned)uni((int)mv.y) << 32) | (unsigned)uni((int)mv.x);
            avHi = ((unsigned long long)(unsigned)uni((int)mv.w) << 32) | (unsigned)uni((int)mv.z);
            asm volatile("s_waitcnt lgkmcnt(0)" ::: "memory");
        }
        if (avLo) { const int b = __builtin_ctzll(avLo); avLo &= avLo - 1; isLo |= 1ull << b; last_kt = b; }
        else if (avHi) { const int b = __builtin_ctzll(avHi); avHi &= avHi - 1; isHi |= 1ull << b; last_kt = 64 + b; }
        else { valid = false; return last_kt; }
        ++n_issued; valid = true; return last_kt;
    };

    bool v0, v1, vn;
    int k0 = pick(true, v0);
    int k1 = pick(false, v1);
    STAMP(1);
    {
        const char* a0 = xb + (size_t)k0 * BLK; const char* b0 = wb + (size_t)k0 * BLK;
        const char* a1 = xb + (size_t)k1 * BLK; const char* b1 = wb + (size_t)k1 * BLK;
        STAGE(SB(0, 0), b0); STAGE(SA(0, 0), a0); STAGE(SB(0, 1), b0 + HTB); STAGE(SA(0, 1), a0 + HTB);
        if (wr == 1) BAR;
        WAIT_V(4); BAR;
        STAGE(SB(1, 0), b1); STAGE(SA(1, 0), a1); STAGE(SB(1, 1), b1 + HTB);
        WAIT_V(6); BAR;
    }
    int done = 0;
#define DG_DUMMY do {} while (0)
#define DG_PROG do {} while (0)
    const char* apend = xb + (size_t)k1 * BLK + HTB;
#define HALF(b, o, vcur, voth) do { \
        const int kn = pick(!(vcur) && !(voth), vn); \
        const char* an = xb + (size_t)kn * BLK; const char* bn = wb + (size_t)kn * BLK; \
        if (vcur) { \
            LDB(B0, b, 0); SCHED; LDA(At, b, 0); STAGE(SA(o, 1), apend); \
            WAIT_L(8); BAR; WAIT_L(0); MMA(0, 0, At, B0); BAR; SCHED; \
            LDB(B1, b, 1); STAGE(SB(b, 0), bn); \
            BAR; WAIT_L(0); MMA(0, 1, At, B1); BAR; \
            LDA(At, b, 1); STAGE(SA(b, 0), an); \
            BAR; WAIT_L(0); MMA(1, 0, At, B0); BAR; SCHED; \
            STAGE(SB(b, 1), bn + HTB); \
            WAIT_V(6); BAR; MMA(1, 1, At, B1); BAR; \
            ++done; \
        } else { \
            STAGE(SA(o, 1), apend); BAR; BAR; \
            STAGE(SB(b, 0), bn); BAR; BAR; \
            STAGE(SA(b, 0), an); BAR; BAR; \
            STAGE(SB(b, 1), bn + HTB); WAIT_V(6); BAR; BAR; \
            DG_DUMMY; \
        } \
        DG_PROG; \
        apend = an + HTB; vcur = vn; } while (0)
    for (;;) {
        HALF(0, 1, v0, v1);
        if (done >= NKT) break;
        HALF(1, 0, v1, v0);
        if (done >= NKT) break;
    }
    WAIT_V(0);
    if (wr == 0) BAR;
    BAR;
    STAMP(2);

    const bool bad = (unsigned)uni((int)ld_agt(tmo)) == call;
    const int grow = row_base(P.rank, ti), owner = grow >> 10;
    char* obase = (char*)P.out[owner] + ((size_t)(grow & 1023) * NG + (size_t)((P.rank >> 2) * 4096 + tj * 256)) * 4;
    const float qnan = __builtin_nanf("");
#pragma unroll
    for (int ai = 0; ai < 2; ++ai) {
#pragma unroll
        for (int m = 0; m < 4; ++m) {
            const int r = wr * 64 + m * 16 + fr;
#pragma unroll
            for (int bj = 0; bj < 2; ++bj)
#pragma unroll
                for (int n = 0; n < 2; ++n) {
                    f32x4 y = acc[ai][bj][m][n];
                    y.x = silu(y.x); y.y = silu(y.y); y.z = silu(y.z); y.w = silu(y.w);
                    if (bad) y = (f32x4){qnan, qnan, qnan, qnan};
                    const int ch = bj * 32 + wc * 8 + n * 4 + fq;
                    *(LAS f32x4*)(lds + r * 1024 + ((ch ^ (r & 7)) << 4)) = y;
                }
        }
        WAIT_L(0); BAR; asm volatile("" ::: "memory");
#pragma unroll
        for (int q = 0; q < 16; ++q) {
            const int r = wid * 16 + q;
            const u32x4 y = *(const LAS u32x4*)(lds + r * 1024 + ((lane ^ (r & 7)) << 4));
            st16_wt(obase + (size_t)(ai * 128 + r) * (NG * 4) + lane * 16, y);
        }
        WAIT_L(0); asm volatile("" ::: "memory"); BAR; asm volatile("" ::: "memory");
    }
    if (owner != P.rank) {
        WAIT_V(0); BAR;
        if (tid == 0) st_sys((unsigned*)(P.ws[owner] + WS_FLO) + (P.rank & 3) * 32 + tj, call);
    }
    STAMP(3);
#undef SA
#undef SB
#undef STAGE
#undef LDA
#undef LDB
#undef MMA
#undef HALF
}

__global__ void __launch_bounds__(512, 2) fused(Params P) {
    extern __shared__ __attribute__((aligned(16))) unsigned char shm[];
    LAS unsigned char* lds = (LAS unsigned char*)shm;
    char* ws = P.ws[P.rank];
    const unsigned call = (unsigned)uni((int)ld_agt((const unsigned*)(ws + WS_DONE))) / (unsigned)GRID + 1u;
    const int b = blockIdx.x;
    STAMP(0);
    if (b >= NPUSH_PAD) {
        const int t = b - NPUSH_PAD, xcd = t & 7, w = t >> 3;
        gemm_tile(P, lds, w & 7, xcd * 2 + (w >> 3), call);
    } else if (b < NPUSH) {
        int dest, v, g;
        if (b < 7 * GL) { const int d = b % 7; dest = (P.rank + 1 + d) & 7; v = b / 7; g = GL; }
        else { dest = P.rank; v = b - 7 * GL; g = GS; }
        pusher(P, lds, dest, v, g, call);
        STAMP(2);
        if (b == NPUSH - 1) {
            if (threadIdx.x < 64) {
                const int lane = threadIdx.x;
                const unsigned* f = (const unsigned*)(ws + WS_FLO) + (lane >> 4) * 32 + (lane & 15);
                unsigned* tmo = (unsigned*)(ws + WS_TMO);
                const unsigned long long t0 = rtc();
                for (;;) {
                    if (__all(ld_sys(f) >= call)) break;
                    if ((unsigned)uni((int)ld_agt(tmo)) == call || rtc() - t0 > DEADLINE) { if (lane == 0) st_agt(tmo, call); break; }
                    __builtin_amdgcn_s_sleep(2);
                }
                if ((unsigned)uni((int)ld_agt(tmo)) == call && lane == 0)
                    P.out[P.rank][(size_t)(P.rank >> 2) * 4096] = __builtin_nanf("");
            }
            STAMP(3);
        }
    }
    __syncthreads();
    if (threadIdx.x == 0) __hip_atomic_fetch_add((gu32*)(ws + WS_DONE), 1u, __ATOMIC_RELAXED, __HIP_MEMORY_SCOPE_AGENT);
}

}

extern "C" void kernel_launch(int rank, int nranks, ncclComm_t comm, hipStream_t stream,
                              void* const* d_in, const int* in_sizes, int n_in,
                              void* d_out, int out_size,
                              const int* in_dtypes, int out_dtype,
                              void* const* d_out_all,
                              void* const* d_ws_all, size_t ws_bytes) {
    using namespace gk;
    (void)comm; (void)in_sizes; (void)n_in; (void)d_out; (void)out_size; (void)in_dtypes; (void)out_dtype;
    if (nranks != NR || ws_bytes < WS_END || rank < 0 || rank >= NR) { fprintf(stderr, "unsupported configuration\n"); return; }
    static bool attr_set[64] = {};
    int dev = 0; HIP_CHECK(hipGetDevice(&dev));
    if (dev >= 0 && dev < 64 && !attr_set[dev]) {
        HIP_CHECK(hipFuncSetAttribute((const void*)fused, hipFuncAttributeMaxDynamicSharedMemorySize, LDS_BYTES));
        attr_set[dev] = true;
    }
    Params P;
    P.x = (const float*)d_in[0]; P.w = (const float*)d_in[1];
    for (int p = 0; p < NR; ++p) { P.ws[p] = (char*)d_ws_all[p]; P.out[p] = (float*)d_out_all[p]; }
    P.rank = rank; P.pad = 0;
    hipLaunchKernelGGL(fused, dim3(GRID), dim3(512), LDS_BYTES, stream, P);
    HIP_CHECK(hipGetLastError());
}
```

```cpp
#include <hip/hip_runtime.h>
#include <rccl/rccl.h>
#include <cstdio>

#define HIP_CHECK(expr) do { hipError_t _e = (expr); if (_e != hipSuccess) { \
    fprintf(stderr, "HIP error %s at %s:%d\n", hipGetErrorString(_e), __FILE__, __LINE__); \
    return; } } while (0)

namespace gk {

#define LAS __attribute__((address_space(3)))
#define GAS __attribute__((address_space(1)))
typedef _Float16 h16x8 __attribute__((ext_vector_type(8)));
typedef _Float16 h16x2 __attribute__((ext_vector_type(2)));
typedef float f32x4 __attribute__((ext_vector_type(4)));
typedef unsigned u32x4 __attribute__((ext_vector_type(4)));
typedef GAS unsigned gu32;

constexpr int NR = 8;
constexpr int NG = 8192;
constexpr int KS = 1024;
constexpr int NKT = 128;
constexpr int NRT = 8, NCT = 16;
constexpr int BLK = 32768;
constexpr int HTB = 16384;

constexpr size_t WS_DONE = 0;
constexpr size_t WS_TMO = 128;
constexpr size_t WS_FLX = 4096;
constexpr size_t WS_FLW = WS_FLX + NRT * NR * 128;
constexpr size_t WS_FLO = WS_FLW + NCT * NR * 128;
constexpr size_t WS_X = 1u << 20;
constexpr size_t XBUF = (size_t)NRT * NKT * BLK;
constexpr size_t WS_W = WS_X + 2 * XBUF;
constexpr size_t WBUF = (size_t)NCT * NKT * BLK;
constexpr size_t WS_END = WS_W + 2 * WBUF;

constexpr float W_SCALE = 32.0f;
constexpr unsigned long long DEADLINE = 150000000ull;

constexpr int PG = 8;
constexpr int GRID = 256;
constexpr int LDS_MAIN = 9 * HTB;
constexpr int LDS_FLG = LDS_MAIN;
constexpr int LDS_CTL = LDS_MAIN + 4096;
constexpr int LDS_BYTES = LDS_MAIN + 4096 + 256;

constexpr int RELAY_PAD = 16;
__constant__ const unsigned short RELAY_SET[64] = {0x0,0x40,0x80,0xc0,0x31,0x71,0xb1,0xf1,0x22,0x62,0xa2,0xe2,0x13,0x53,0xffff,0xffff,0x10,0x50,0x90,0xd0,0x1,0x41,0x81,0xc1,0x32,0x72,0xb2,0xf2,0x23,0x63,0xffff,0xffff,0x20,0x60,0xa0,0xe0,0x11,0x51,0x91,0xd1,0x2,0x42,0x82,0xc2,0x33,0x73,0xffff,0xffff,0x30,0x70,0xb0,0xf0,0x21,0x61,0xa1,0xe1,0x12,0x52,0x92,0xd2,0x3,0x43,0xffff,0xffff};
constexpr unsigned short FIRST_ENT[4][64] = {{0x1,0x10,0x14,0x18,0x1c,0x5,0x41,0x50,0x1,0x10,0x14,0x18,0x1c,0x5,0x41,0x50,0x1,0x10,0x14,0x18,0x1c,0x5,0x41,0x50,0x10,0x14,0x18,0x1c,0x1,0x5,0x9,0xd,0x1,0x11,0x21,0x31,0x4d,0x5d,0x6d,0x7d,0x5,0x15,0x25,0x35,0x41,0x51,0x61,0x71,0x9,0x19,0x29,0x39,0x45,0x55,0x65,0x75,0xd,0x1d,0x2d,0x3d,0x49,0x59,0x69,0x79},
    {0x1,0x1c,0x10,0x14,0x18,0x5,0x41,0x5c,0x1,0x1c,0x10,0x14,0x18,0x5,0x41,0x5c,0x1,0x1c,0x10,0x14,0x18,0x5,0x41,0x5c,0x10,0x14,0x18,0x1c,0x1,0x5,0x9,0xd,0x1,0x11,0x21,0x31,0x4d,0x5d,0x6d,0x7d,0x5,0x15,0x25,0x35,0x41,0x51,0x61,0x71,0x9,0x19,0x29,0x39,0x45,0x55,0x65,0x75,0xd,0x1d,0x2d,0x3d,0x49,0x59,0x69,0x79},
    {0x1,0x18,0x1c,0x10,0x14,0x5,0x41,0x58,0x1,0x18,0x1c,0x10,0x14,0x5,0x41,0x58,0x1,0x18,0x1c,0x10,0x14,0x5,0x41,0x58,0x10,0x14,0x18,0x1c,0x1,0x5,0x9,0xd,0x1,0x11,0x21,0x31,0x4d,0x5d,0x6d,0x7d,0x5,0x15,0x25,0x35,0x41,0x51,0x61,0x71,0x9,0x19,0x29,0x39,0x45,0x55,0x65,0x75,0xd,0x1d,0x2d,0x3d,0x49,0x59,0x69,0x79},
    {0x1,0x14,0x18,0x1c,0x10,0x5,0x41,0x54,0x1,0x14,0x18,0x1c,0x10,0x5,0x41,0x54,0x1,0x14,0x18,0x1c,0x10,0x5,0x41,0x54,0x10,0x14,0x18,0x1c,0x1,0x5,0x9,0xd,0x1,0x11,0x21,0x31,0x4d,0x5d,0x6d,0x7d,0x5,0x15,0x25,0x35,0x41,0x51,0x61,0x71,0x9,0x19,0x29,0x39,0x45,0x55,0x65,0x75,0xd,0x1d,0x2d,0x3d,0x49,0x59,0x69,0x79}};
constexpr int TAB_INTRA_LEN = 440, TAB_INTRA_WORDS = 220;
__constant__ const unsigned TAB_INTRA[880] = {
    0x00100001,0x00180014,0x0005001c,0x00500041,0x00580054,0x0009005c,0x000d0045,0x00900081,0x00980094,0x0049009c,0x00850011,0x0015004d,
    0x00d000c1,0x00d800d4,0x008900dc,0x00190051,0x008d00c5,0x01010055,0x0110001d,0x01180114,0x00c9011c,0x00910002,0x01050059,0x00cd0021,
    0x009500c6,0x005d0141,0x01540150,0x015c0158,0x00250109,0x008a00d1,0x01450099,0x010d0061,0x00d50029,0x0181004e,0x0190009d,0x01980194,
    0x0149019c,0x01110065,0x00d9002d,0x01850012,0x014d00a1,0x01150069,0x01c10031,0x01d000dd,0x01d801d4,0x00d601dc,0x00a50189,0x01510402,
    0x0119006d,0x01c50035,0x009a00e1,0x00a9018d,0x015504c6,0x02010071,0x0210011d,0x02180214,0x0039021c,0x00e501c9,0x0191005e,0x048a00ad,
    0x00750159,0x01210205,0x01cd003d,0x002200e9,0x00b10195,0x044e0241,0x0079015d,0x02540250,0x025c0258,0x01250209,0x00ed01d1,0x00b50199,
    0x04120245,0x007d0161,0x0129020d,0x00f101d5,0x00aa0281,0x00b9019d,0x02940290,0x029c0298,0x04d60249,0x02110165,0x0802012d,0x00f501d9,
    0x006e0285,0x00bd01a1,0x049a024d,0x02150169,0x08c60131,0x01dd02c1,0x02d000f9,0x02d802d4,0x028902dc,0x01a50032,0x045e0251,0x0219016d,
    0x088a0135,0x01e102c5,0x028d00fd,0x025501a9,0x01710422,0x021d0301,0x084e0139,0x03140310,0x031c0318,0x01e502c9,0x00ba0291,0x025901ad,
    0x03050175,0x013d0221,0x02cd0812,0x029501e9,0x01b1007e,0x025d0341,0x017904aa,0x03540350,0x035c0358,0x02250309,0x02d108d6,0x0c0201ed,
    0x01b50299,0x02610345,0x017d046e,0x0229030d,0x02d5089a,0x038101f1,0x029d0cc6,0x039001b9,0x03980394,0x0349039c,0x04320265,0x022d0311,
    0x02d9085e,0x038501f5,0x02a10c8a,0x034d01bd,0x03150269,0x03c10231,0x02dd0822,0x03d001f9,0x03d803d4,0x038903dc,0x02a50c4e,0x026d0351,
    0x031904ba,0x03c50235,0x01fd02e1,0x0c12038d,0x035502a9,0x047e0271,0x0239031d,0x08aa03c9,0x000002e5,0x0cd60391,0x035902ad,0x02750004,
    0x023d0321,0x086e03cd,0x02e90008,0x0c9a0395,0x000c02b1,0x0279035d,0x00400325,0x083203d1,0x039902ed,0x02b50c5e,0x03610044,0x0048027d,
    0x03d50329,0x004c02f1,0x0c22039d,0x036502b9,0x032d0080,0x08ba03d9,0x008402f5,0x02bd03a1,0x00880369,0x03dd0331,0x008c087e,0x03a502f9,
    0x00c00caa,0x0335036d,0x03e100c4,0x03a902fd,0x00c80c6e,0x03390371,0x03e500cc,0x010003ad,0x03750c32,0x033d0104,0x010803e9,0x037903b1,
    0x03ed010c,0x03b50140,0x037d0cba,0x01480144,0x03b903f1,0x014c0c7e,0x03f50180,0x018403bd,0x03f90188,0x01c0018c,0x01c403fd,0x01cc01c8,
    0x02040200,0x020c0208,0x02440240,0x024c0248,0x02840280,0x028c0288,0x02c402c0,0x02cc02c8,0x03040300,0x030c0308,0x03440340,0x034c0348,
    0x03840380,0x038c0388,0x03c403c0,0x03cc03c8,0x001c0001,0x00140010,0x00050018,0x005c0041,0x00540050,0x00090058,0x000d0045,0x009c0081,
    0x00940090,0x00490098,0x00850011,0x0015004d,0x00dc00c1,0x00d400d0,0x008900d8,0x00190051,0x008d00c5,0x01010055,0x011c001d,0x01140110,
    0x00c90118,0x00910042,0x01050059,0x00cd0021,0x00950006,0x005d0141,0x0150015c,0x01580154,0x00250109,0x00ca00d1,0x01450099,0x010d0061,
    0x00d50029,0x0181008e,0x019c009d,0x01940190,0x01490198,0x01110065,0x00d9002d,0x01850052,0x014d00a1,0x01150069,0x01c10031,0x01dc00dd,
    0x01d401d0,0x001601d8,0x00a50189,0x01510442,0x0119006d,0x01c50035,0x00da00e1,0x00a9018d,0x01550406,0x02010071,0x021c011d,0x02140210,
    0x00390218,0x00e501c9,0x0191009e,0x04ca00ad,0x00750159,0x01210205,0x01cd003d,0x006200e9,0x00b10195,0x048e0241,0x0079015d,0x0250025c,
    0x02580254,0x01250209,0x00ed01d1,0x01990026,0x024500b5,0x01610452,0x020d007d,0x01d50129,0x028100f1,0x00b9019d,0x0290029c,0x02980294,
    0x04160249,0x02110165,0x0842012d,0x00f501d9,0x00ae0285,0x00bd01a1,0x04da024d,0x02150169,0x08060131,0x01dd02c1,0x02dc00f9,0x02d402d0,
    0x028902d8,0x01a50072,0x049e0251,0x0219016d,0x08ca0135,0x01e102c5,0x028d00fd,0x01a90036,0x04620255,0x03010171,0x0139021d,0x031c088e,
    0x03140310,0x02c90318,0x029101e5,0x025901ad,0x01750426,0x02210305,0x0852013d,0x01e902cd,0x00be0295,0x034101b1,0x0179025d,0x0350035c,
    0x03580354,0x02250309,0x02d10816,0x0c4201ed,0x01b50299,0x02610345,0x017d04ae,0x0229030d,0x02d508da,0x038101f1,0x029d0c06,0x039c01b9,
    0x03940390,0x03490398,0x04720265,0x022d0311,0x02d9089e,0x038501f5,0x02a10cca,0x034d01bd,0x04360269,0x02310315,0x086203c1,0x01f902dd,
    0x03d003dc,0x03d803d4,0x0c8e0389,0x035102a5,0x0319026d,0x03c50235,0x02e10826,0x038d01fd,0x02a90c52,0x02710355,0x031d04be,0x03c90239,
    0x000c02e5,0x0c160391,0x035902ad,0x02750000,0x023d0321,0x08ae03cd,0x02e90004,0x0cda0395,0x000802b1,0x0279035d,0x004c0325,0x087203d1,
    0x039902ed,0x02b50c9e,0x03610040,0x0044027d,0x03d50329,0x02f10836,0x039d0048,0x02b90c62,0x008c0365,0x03d9032d,0x008002f5,0x0c2603a1,
    0x036902bd,0x03310084,0x08be03dd,0x02f90088,0x00cc03a5,0x0335036d,0x03e100c0,0x03a902fd,0x00c40cae,0x03390371,0x03e500c8,0x010c03ad,
    0x03750c72,0x033d0100,0x010403e9,0x0c3603b1,0x01080379,0x014c03ed,0x037d03b5,0x01440140,0x03b903f1,0x01480cbe,0x03f5018c,0x018003bd,
    0x03f90184,0x01cc0188,0x01c003fd,0x01c801c4,0x0200020c,0x02080204,0x0240024c,0x02480244,0x0280028c,0x02880284,0x02c002cc,0x02c802c4,
    0x0300030c,0x03080304,0x0340034c,0x03480344,0x0380038c,0x03880384,0x03c003cc,0x03c803c4,0x00180001,0x0010001c,0x00050014,0x00580041,
    0x0050005c,0x00090054,0x000d0045,0x00980081,0x0090009c,0x00490094,0x00850011,0x0015004d,0x00d800c1,0x00d000dc,0x008900d4,0x00190051,
    0x008d00c5,0x01010055,0x0118001d,0x0110011c,0x00c90114,0x00910082,0x01050059,0x00cd0021,0x00950046,0x005d0141,0x015c0158,0x01540150,
    0x00250109,0x000a00d1,0x01450099,0x010d0061,0x00d50029,0x018100ce,0x0198009d,0x0190019c,0x01490194,0x01110065,0x00d9002d,0x01850092,
    0x014d00a1,0x01150069,0x01c10031,0x01d800dd,0x01d001dc,0x005601d4,0x00a50189,0x01510482,0x0119006d,0x01c50035,0x001a00e1,0x00a9018d,
    0x01550446,0x02010071,0x0218011d,0x0210021c,0x00390214,0x00e501c9,0x019100de,0x040a00ad,0x00750159,0x01210205,0x01cd003d,0x00a200e9,
    0x00b10195,0x04ce0241,0x0079015d,0x025c0258,0x02540250,0x01250209,0x00ed01d1,0x01990066,0x024500b5,0x01610492,0x020d007d,0x01d50129,
    0x028100f1,0x019d002a,0x029800b9,0x0290029c,0x02490294,0x01650456,0x012d0211,0x01d90882,0x028500f5,0x00bd01a1,0x041a024d,0x02150169,
    0x08460131,0x01dd02c1,0x02d800f9,0x02d002dc,0x028902d4,0x01a500b2,0x04de0251,0x0219016d,0x080a0135,0x01e102c5,0x028d00fd,0x01a90076,
    0x04a20255,0x03010171,0x0139021d,0x031808ce,0x0310031c,0x02c90314,0x029101e5,0x01ad003a,0x04660259,0x03050175,0x013d0221,0x02cd0892,
    0x029501e9,0x034101b1,0x042a025d,0x03580179,0x0350035c,0x03090354,0x08560225,0x01ed02d1,0x02990c82,0x034501b5,0x017d0261,0x0229030d,
    0x02d5081a,0x038101f1,0x029d0c46,0x039801b9,0x0390039c,0x03490394,0x04b20265,0x022d0311,0x02d908de,0x038501f5,0x02a10c0a,0x034d01bd,
    0x04760269,0x02310315,0x08a203c1,0x01f902dd,0x03dc03d8,0x03d403d0,0x0cce0389,0x035102a5,0x043a026d,0x02350319,0x086603c5,0x01fd02e1,
    0x0c92038d,0x035502a9,0x031d0271,0x03c90239,0x02e5082a,0x03910008,0x02ad0c56,0x000c0359,0x03210275,0x03cd023d,0x02e90000,0x0c1a0395,
    0x000402b1,0x0279035d,0x00480325,0x08b203d1,0x039902ed,0x02b50cde,0x0361004c,0x0040027d,0x03d50329,0x02f10876,0x039d0044,0x02b90ca2,
    0x00880365,0x03d9032d,0x02f5083a,0x03a1008c,0x02bd0c66,0x00800369,0x03dd0331,0x02f90084,0x0c2a03a5,0x036d00c8,0x00cc0335,0x02fd03e1,
    0x00c003a9,0x03390371,0x03e500c4,0x010803ad,0x03750cb2,0x033d010c,0x010003e9,0x0c7603b1,0x01040379,0x014803ed,0x0c3a03b5,0x014c037d,
    0x03f10140,0x014403b9,0x03f50188,0x018c03bd,0x03f90180,0x01c80184,0x01cc03fd,0x01c401c0,0x020c0208,0x02040200,0x024c0248,0x02440240,
    0x028c0288,0x02840280,0x02cc02c8,0x02c402c0,0x030c0308,0x03040300,0x034c0348,0x03440340,0x038c0388,0x03840380,0x03cc03c8,0x03c403c0,
    0x00140001,0x001c0018,0x00050010,0x00540041,0x005c0058,0x00090050,0x000d0045,0x00940081,0x009c0098,0x00490090,0x00850011,0x0015004d,
    0x00d400c1,0x00dc00d8,0x008900d0,0x00190051,0x008d00c5,0x01010055,0x0114001d,0x011c0118,0x00c90110,0x009100c2,0x01050059,0x00cd0021,
    0x00950086,0x005d0141,0x01580154,0x0150015c,0x00250109,0x004a00d1,0x01450099,0x010d0061,0x00d50029,0x0181000e,0x0194009d,0x019c0198,
    0x01490190,0x01110065,0x00d9002d,0x018500d2,0x014d00a1,0x01150069,0x01c10031,0x01d400dd,0x01dc01d8,0x009601d0,0x00a50189,0x015104c2,
    0x0119006d,0x01c50035,0x005a00e1,0x00a9018d,0x01550486,0x02010071,0x0214011d,0x021c0218,0x00390210,0x00e501c9,0x0191001e,0x044a00ad,
    0x00750159,0x01210205,0x01cd003d,0x019500e9,0x024100b1,0x015d040e,0x02540079,0x025c0258,0x02090250,0x01d10125,0x00a600ed,0x00b50199,
    0x04d20245,0x007d0161,0x0129020d,0x00f101d5,0x006a0281,0x00b9019d,0x02980294,0x0290029c,0x04960249,0x02110165,0x08c2012d,0x00f501d9,
    0x002e0285,0x00bd01a1,0x045a024d,0x02150169,0x08860131,0x01dd02c1,0x02d400f9,0x02dc02d8,0x028902d0,0x025101a5,0x016d041e,0x01350219,
    0x02c5084a,0x00fd01e1,0x00b6028d,0x025501a9,0x03010171,0x0139021d,0x0314080e,0x031c0318,0x02c90310,0x029101e5,0x01ad007a,0x04a60259,
    0x03050175,0x013d0221,0x02cd08d2,0x029501e9,0x01b1003e,0x025d0341,0x0179046a,0x03580354,0x0350035c,0x02250309,0x02d10896,0x0cc201ed,
    0x01b50299,0x02610345,0x017d042e,0x0229030d,0x02d5085a,0x038101f1,0x029d0c86,0x039401b9,0x039c0398,0x03490390,0x03110265,0x081e022d,
    0x01f502d9,0x0c4a0385,0x01bd02a1,0x0269034d,0x031504b6,0x03c10231,0x01f902dd,0x03d803d4,0x03d003dc,0x0c0e0389,0x035102a5,0x047a026d,
    0x02350319,0x08a603c5,0x01fd02e1,0x0cd2038d,0x035502a9,0x043e0271,0x0239031d,0x086a03c9,0x000402e5,0x0c960391,0x035902ad,0x02750008,
    0x023d0321,0x082e03cd,0x02e9000c,0x0c5a0395,0x000002b1,0x0279035d,0x00440325,0x02ed03d1,0x0c1e0399,0x004802b5,0x027d0361,0x0329004c,
    0x08b603d5,0x004002f1,0x02b9039d,0x00840365,0x03d9032d,0x02f5087a,0x03a10088,0x02bd0ca6,0x008c0369,0x03dd0331,0x0080083e,0x03a502f9,
    0x00c40c6a,0x0335036d,0x03e100c8,0x03a902fd,0x00cc0c2e,0x03390371,0x03e500c0,0x010403ad,0x01080375,0x03e9033d,0x03b1010c,0x03790cb6,
    0x03ed0100,0x03b50144,0x037d0c7a,0x014c0148,0x03b903f1,0x01400c3e,0x03f50184,0x018803bd,0x03f9018c,0x01c40180,0x01c803fd,0x01c001cc,
    0x02080204,0x0200020c,0x02480244,0x0240024c,0x02880284,0x0280028c,0x02c802c4,0x02c002cc,0x03080304,0x0300030c,0x03480344,0x0340034c,
    0x03880384,0x0380038c,0x03c803c4,0x03c003cc,
};
constexpr int TAB_INTER_LEN = 342, TAB_INTER_WORDS = 171;
__constant__ const unsigned TAB_INTER[684] = {
    0x00110001,0x00310021,0x005d004d,0x007d006d,0x00990089,0x00b900a9,0x00d500c5,0x01050101,0x010d0109,0x01150111,0x011d0119,0x00e500e1,
    0x00e90141,0x00ed0145,0x00f10149,0x00f5014d,0x00f90151,0x00fd0155,0x01210159,0x0125015d,0x01810129,0x0185012d,0x01890131,0x018d0135,
    0x01910139,0x0195013d,0x01990161,0x019d0165,0x01c10169,0x01c5016d,0x01c90171,0x01cd0175,0x01d10179,0x01d5017d,0x01d901a1,0x01dd01a5,
    0x01ad01a9,0x01b10201,0x01b50205,0x01b90209,0x01bd020d,0x01e10211,0x01e50215,0x01e90219,0x01ed021d,0x024101f1,0x024501f5,0x001001f9,
    0x00180014,0x0249001c,0x024d01fd,0x02510221,0x02550225,0x02590229,0x00540050,0x005c0058,0x025d022d,0x02810231,0x02850235,0x00900239,
    0x00980094,0x0289009c,0x028d023d,0x02910261,0x02950265,0x00d00269,0x00d800d4,0x029900dc,0x029d026d,0x02750271,0x027902c1,0x011002c5,
    0x01180114,0x027d011c,0x02a102c9,0x02a502cd,0x02a902d1,0x015002d5,0x015402ad,0x015c0158,0x02b102d9,0x02b502dd,0x030102b9,0x019002bd,
    0x01980194,0x0305019c,0x030902e1,0x030d02e5,0x031102e9,0x031502ed,0x01d401d0,0x01dc01d8,0x031902f1,0x031d02f5,0x02fd02f9,0x02100341,
    0x02180214,0x0345021c,0x03490321,0x034d0325,0x03510329,0x0250032d,0x02580254,0x0355025c,0x03590331,0x035d0335,0x033d0339,0x02900381,
    0x02980294,0x0361029c,0x03650385,0x03690389,0x036d038d,0x03710391,0x02d402d0,0x02dc02d8,0x03750395,0x03790399,0x037d039d,0x031003a1,
    0x03180314,0x03c1031c,0x03c503a5,0x03c903a9,0x03cd03ad,0x03d103b1,0x03540350,0x035c0358,0x03d503b5,0x03d903b9,0x03dd03bd,0x039003e1,
    0x03980394,0x03e5039c,0x03ed03e9,0x03d003f1,0x03d803d4,0x03f503dc,0x000003f9,0x03fd0004,0x000c0008,0x00440040,0x004c0048,0x00840080,
    0x008c0088,0x00c400c0,0x00cc00c8,0x01040100,0x010c0108,0x01440140,0x014c0148,0x01840180,0x018c0188,0x01c401c0,0x01cc01c8,0x02040200,
    0x020c0208,0x02440240,0x024c0248,0x02840280,0x028c0288,0x02c402c0,0x02cc02c8,0x03040300,0x030c0308,0x03440340,0x034c0348,0x03840380,
    0x038c0388,0x03c403c0,0x03cc03c8,0x00150005,0x00350025,0x00510041,0x00710061,0x009d008d,0x00bd00ad,0x00d900c9,0x01050101,0x010d0109,
    0x01150111,0x011d0119,0x00e500e1,0x00e90141,0x00ed0145,0x00f10149,0x00f5014d,0x00f90151,0x00fd0155,0x01210159,0x0125015d,0x01810129,
    0x0185012d,0x01890131,0x018d0135,0x01910139,0x0195013d,0x01990161,0x019d0165,0x01c10169,0x01c5016d,0x01c90171,0x01cd0175,0x01d10179,
    0x01d5017d,0x01d901a1,0x01dd01a5,0x01ad01a9,0x01b10201,0x01b50205,0x01b90209,0x01bd020d,0x01e10211,0x01e50215,0x01e90219,0x01ed021d,
    0x024101f1,0x024501f5,0x001c01f9,0x00140010,0x02490018,0x024d01fd,0x02510221,0x02550225,0x02590229,0x0050005c,0x00580054,0x025d022d,
    0x02810231,0x02850235,0x009c0239,0x00940090,0x02890098,0x028d023d,0x02910261,0x02950265,0x00dc0269,0x00d400d0,0x029900d8,0x029d026d,
    0x02750271,0x027902c1,0x011c02c5,0x01140110,0x027d0118,0x02a102c9,0x02a502cd,0x02a902d1,0x015c02d5,0x015002ad,0x01580154,0x02b102d9,
    0x02b502dd,0x030102b9,0x019c02bd,0x01940190,0x03050198,0x030902e1,0x030d02e5,0x031102e9,0x031502ed,0x01d001dc,0x01d801d4,0x031902f1,
    0x031d02f5,0x02fd02f9,0x021c0341,0x02140210,0x03450218,0x03490321,0x034d0325,0x03510329,0x025c032d,0x02540250,0x03550258,0x03590331,
    0x035d0335,0x033d0339,0x029c0381,0x02940290,0x03610298,0x03650385,0x03690389,0x036d038d,0x03710391,0x02d002dc,0x02d802d4,0x03750395,
    0x03790399,0x037d039d,0x031c03a1,0x03140310,0x03c10318,0x03c503a5,0x03c903a9,0x03cd03ad,0x03d103b1,0x0350035c,0x03580354,0x03d503b5,
    0x03d903b9,0x03dd03bd,0x039c03e1,0x03940390,0x03e50398,0x03ed03e9,0x03dc03f1,0x03d403d0,0x03f503d8,0x000c03f9,0x03fd0000,0x00080004,
    0x0040004c,0x00480044,0x0080008c,0x00880084,0x00c000cc,0x00c800c4,0x0100010c,0x01080104,0x0140014c,0x01480144,0x0180018c,0x01880184,
    0x01c001cc,0x01c801c4,0x0200020c,0x02080204,0x0240024c,0x02480244,0x0280028c,0x02880284,0x02c002cc,0x02c802c4,0x0300030c,0x03080304,
    0x0340034c,0x03480344,0x0380038c,0x03880384,0x03c003cc,0x03c803c4,0x00190009,0x00390029,0x00550045,0x00750065,0x00910081,0x00b100a1,
    0x00dd00cd,0x01050101,0x010d0109,0x01150111,0x011d0119,0x00e500e1,0x00e90141,0x00ed0145,0x00f10149,0x00f5014d,0x00f90151,0x00fd0155,
    0x01210159,0x0125015d,0x01810129,0x0185012d,0x01890131,0x018d0135,0x01910139,0x0195013d,0x01990161,0x019d0165,0x01c10169,0x01c5016d,
    0x01c90171,0x01cd0175,0x01d10179,0x01d5017d,0x01d901a1,0x01dd01a5,0x01ad01a9,0x01b10201,0x01b50205,0x01b90209,0x01bd020d,0x01e10211,
    0x01e50215,0x01e90219,0x01ed021d,0x024101f1,0x024501f5,0x001801f9,0x0010001c,0x02490014,0x024d01fd,0x02510221,0x02550225,0x02590229,
    0x005c0058,0x00540050,0x025d022d,0x02810231,0x02850235,0x00980239,0x0090009c,0x02890094,0x028d023d,0x02910261,0x02950265,0x00d80269,
    0x00d000dc,0x029900d4,0x029d026d,0x02750271,0x027902c1,0x011802c5,0x0110011c,0x027d0114,0x02a102c9,0x02a502cd,0x02a902d1,0x015802d5,
    0x015c02ad,0x01540150,0x02b102d9,0x02b502dd,0x030102b9,0x019802bd,0x0190019c,0x03050194,0x030902e1,0x030d02e5,0x031102e9,0x031502ed,
    0x01dc01d8,0x01d401d0,0x031902f1,0x031d02f5,0x02fd02f9,0x02180341,0x0210021c,0x03450214,0x03490321,0x034d0325,0x03510329,0x0258032d,
    0x0250025c,0x03550254,0x03590331,0x035d0335,0x033d0339,0x02980381,0x0290029c,0x03610294,0x03650385,0x03690389,0x036d038d,0x03710391,
    0x02dc02d8,0x02d402d0,0x03750395,0x03790399,0x037d039d,0x031803a1,0x0310031c,0x03c10314,0x03c503a5,0x03c903a9,0x03cd03ad,0x03d103b1,
    0x035c0358,0x03540350,0x03d503b5,0x03d903b9,0x03dd03bd,0x039803e1,0x0390039c,0x03e50394,0x03ed03e9,0x03d803f1,0x03d003dc,0x03f503d4,
    0x000803f9,0x03fd000c,0x00040000,0x004c0048,0x00440040,0x008c0088,0x00840080,0x00cc00c8,0x00c400c0,0x010c0108,0x01040100,0x014c0148,
    0x01440140,0x018c0188,0x01840180,0x01cc01c8,0x01c401c0,0x020c0208,0x02040200,0x024c0248,0x02440240,0x028c0288,0x02840280,0x02cc02c8,
    0x02c402c0,0x030c0308,0x03040300,0x034c0348,0x03440340,0x038c0388,0x03840380,0x03cc03c8,0x03c403c0,0x001d000d,0x003d002d,0x00590049,
    0x00790069,0x00950085,0x00b500a5,0x00d100c1,0x01050101,0x010d0109,0x01150111,0x011d0119,0x00e500e1,0x00e90141,0x00ed0145,0x00f10149,
    0x00f5014d,0x00f90151,0x00fd0155,0x01210159,0x0125015d,0x01810129,0x0185012d,0x01890131,0x018d0135,0x01910139,0x0195013d,0x01990161,
    0x019d0165,0x01c10169,0x01c5016d,0x01c90171,0x01cd0175,0x01d10179,0x01d5017d,0x01d901a1,0x01dd01a5,0x01ad01a9,0x01b10201,0x01b50205,
    0x01b90209,0x01bd020d,0x01e10211,0x01e50215,0x01e90219,0x01ed021d,0x024101f1,0x024501f5,0x001401f9,0x001c0018,0x02490010,0x024d01fd,
    0x02510221,0x02550225,0x02590229,0x00580054,0x0050005c,0x025d022d,0x02810231,0x02850235,0x00940239,0x009c0098,0x02890090,0x028d023d,
    0x02910261,0x02950265,0x00d40269,0x00dc00d8,0x029900d0,0x029d026d,0x02750271,0x027902c1,0x011402c5,0x011c0118,0x027d0110,0x02a102c9,
    0x02a502cd,0x02a902d1,0x015402d5,0x015802ad,0x0150015c,0x02b102d9,0x02b502dd,0x030102b9,0x019402bd,0x019c0198,0x03050190,0x030902e1,
    0x030d02e5,0x031102e9,0x031502ed,0x01d801d4,0x01d001dc,0x031902f1,0x031d02f5,0x02fd02f9,0x02140341,0x021c0218,0x03450210,0x03490321,
    0x034d0325,0x03510329,0x0254032d,0x025c0258,0x03550250,0x03590331,0x035d0335,0x033d0339,0x02940381,0x029c0298,0x03610290,0x03650385,
    0x03690389,0x036d038d,0x03710391,0x02d802d4,0x02d002dc,0x03750395,0x03790399,0x037d039d,0x031403a1,0x031c0318,0x03c10310,0x03c503a5,
    0x03c903a9,0x03cd03ad,0x03d103b1,0x03580354,0x0350035c,0x03d503b5,0x03d903b9,0x03dd03bd,0x039403e1,0x039c0398,0x03e50390,0x03ed03e9,
    0x03d403f1,0x03dc03d8,0x03f503d0,0x000403f9,0x03fd0008,0x0000000c,0x00480044,0x0040004c,0x00880084,0x0080008c,0x00c800c4,0x00c000cc,
    0x01080104,0x0100010c,0x01480144,0x0140014c,0x01880184,0x0180018c,0x01c801c4,0x01c001cc,0x02080204,0x0200020c,0x02480244,0x0240024c,
    0x02880284,0x0280028c,0x02c802c4,0x02c002cc,0x03080304,0x0300030c,0x03480344,0x0340034c,0x03880384,0x0380038c,0x03c803c4,0x03c003cc,
};
constexpr int TAB_SELF_LEN = 384, TAB_SELF_WORDS = 192;
__constant__ const unsigned TAB_SELF[192] = {
    0x00140010,0x001c0018,0x00050001,0x000d0009,0x00150011,0x001d0019,0x00250021,0x002d0029,0x00350031,0x003d0039,0x00040000,0x000c0008,
    0x00540050,0x005c0058,0x00450041,0x004d0049,0x00550051,0x005d0059,0x00650061,0x006d0069,0x00750071,0x007d0079,0x00440040,0x004c0048,
    0x00940090,0x009c0098,0x00850081,0x008d0089,0x00950091,0x009d0099,0x00a500a1,0x00ad00a9,0x00b500b1,0x00bd00b9,0x00840080,0x008c0088,
    0x00d400d0,0x00dc00d8,0x00c500c1,0x00cd00c9,0x00d500d1,0x00dd00d9,0x00e500e1,0x00ed00e9,0x00f500f1,0x00fd00f9,0x00c400c0,0x00cc00c8,
    0x01140110,0x011c0118,0x01050101,0x010d0109,0x01150111,0x011d0119,0x01250121,0x012d0129,0x01350131,0x013d0139,0x01040100,0x010c0108,
    0x01540150,0x015c0158,0x01450141,0x014d0149,0x01550151,0x015d0159,0x01650161,0x016d0169,0x01750171,0x017d0179,0x01440140,0x014c0148,
    0x01940190,0x019c0198,0x01850181,0x018d0189,0x01950191,0x019d0199,0x01a501a1,0x01ad01a9,0x01b501b1,0x01bd01b9,0x01840180,0x018c0188,
    0x01d401d0,0x01dc01d8,0x01c501c1,0x01cd01c9,0x01d501d1,0x01dd01d9,0x01e501e1,0x01ed01e9,0x01f501f1,0x01fd01f9,0x01c401c0,0x01cc01c8,
    0x02140210,0x021c0218,0x02050201,0x020d0209,0x02150211,0x021d0219,0x02250221,0x022d0229,0x02350231,0x023d0239,0x02040200,0x020c0208,
    0x02540250,0x025c0258,0x02450241,0x024d0249,0x02550251,0x025d0259,0x02650261,0x026d0269,0x02750271,0x027d0279,0x02440240,0x024c0248,
    0x02940290,0x029c0298,0x02850281,0x028d0289,0x02950291,0x029d0299,0x02a502a1,0x02ad02a9,0x02b502b1,0x02bd02b9,0x02840280,0x028c0288,
    0x02d402d0,0x02dc02d8,0x02c502c1,0x02cd02c9,0x02d502d1,0x02dd02d9,0x02e502e1,0x02ed02e9,0x02f502f1,0x02fd02f9,0x02c402c0,0x02cc02c8,
    0x03140310,0x031c0318,0x03050301,0x030d0309,0x03150311,0x031d0319,0x03250321,0x032d0329,0x03350331,0x033d0339,0x03040300,0x030c0308,
    0x03540350,0x035c0358,0x03450341,0x034d0349,0x03550351,0x035d0359,0x03650361,0x036d0369,0x03750371,0x037d0379,0x03440340,0x034c0348,
    0x03940390,0x039c0398,0x03850381,0x038d0389,0x03950391,0x039d0399,0x03a503a1,0x03ad03a9,0x03b503b1,0x03bd03b9,0x03840380,0x038c0388,
    0x03d403d0,0x03dc03d8,0x03c503c1,0x03cd03c9,0x03d503d1,0x03dd03d9,0x03e503e1,0x03ed03e9,0x03f503f1,0x03fd03f9,0x03c403c0,0x03cc03c8,
};

struct Params {
    const float* x; const float* w;
    char* ws[NR]; float* out[NR];
    int rank; int pad;
    unsigned short first[64];
};

__device__ __forceinline__ unsigned long long rtc() { return __builtin_amdgcn_s_memrealtime(); }
__device__ __forceinline__ unsigned ld_sys(const unsigned* p) { return __hip_atomic_load((const gu32*)p, __ATOMIC_RELAXED, __HIP_MEMORY_SCOPE_SYSTEM); }
__device__ __forceinline__ unsigned ld_agt(const unsigned* p) { return __hip_atomic_load((const gu32*)p, __ATOMIC_RELAXED, __HIP_MEMORY_SCOPE_AGENT); }
__device__ __forceinline__ void st_sys(unsigned* p, unsigned v) { __hip_atomic_store((gu32*)p, v, __ATOMIC_RELAXED, __HIP_MEMORY_SCOPE_SYSTEM); }
__device__ __forceinline__ void st_agt(unsigned* p, unsigned v) { __hip_atomic_store((gu32*)p, v, __ATOMIC_RELAXED, __HIP_MEMORY_SCOPE_AGENT); }
__device__ __forceinline__ void st16_wt(void* p, u32x4 v) { asm volatile("global_store_dwordx4 %0, %1, off sc0 sc1\n\ts_nop 1" :: "v"(p), "v"(v) : "memory"); }
__device__ __forceinline__ unsigned pk2(float a, float b) { h16x2 v; v.x = (_Float16)a; v.y = (_Float16)b; return __builtin_bit_cast(unsigned, v); }
__device__ __forceinline__ int uni(int v) { return __builtin_amdgcn_readfirstlane(v); }
template <class T> __device__ __forceinline__ T* opaque(T* p) { asm volatile("" : "+s"(p)); return p; }
#define WAIT_V(n) asm volatile("s_waitcnt vmcnt(" #n ")" ::: "memory")
#define WAIT_L(n) asm volatile("s_waitcnt lgkmcnt(" #n ")" ::: "memory")
#define BAR __builtin_amdgcn_s_barrier()
#define SCHED __builtin_amdgcn_sched_barrier(0)
#define CFENCE asm volatile("" ::: "memory")
#define WAITN(n) do { const int _n = (n); if (_n >= 10) WAIT_V(10); else if (_n >= 8) WAIT_V(8); else if (_n >= 6) WAIT_V(6); else if (_n >= 4) WAIT_V(4); else if (_n >= 2) WAIT_V(2); else WAIT_V(0); } while (0)

__host__ __device__ __forceinline__ int grp_of(int r) { return (0xB2 >> r) & 1; }
__host__ __device__ __forceinline__ int idx_of(int r) { return (0xF990 >> (2 * r)) & 3; }
__host__ __device__ __forceinline__ int rank_of(int g, int u) { return (0x75416320u >> (16 * g + 4 * u)) & 7; }
__device__ __forceinline__ int row_base(int a, int i) { return i < 4 ? a * 1024 + i * 256 : rank_of(grp_of(a) ^ 1, i - 4) * 1024 + idx_of(a) * 256; }

struct Role { int push_v, ti, tj, half; };
__host__ __device__ __forceinline__ Role role_of(int x, int slot) {
    Role r = {-1, -1, 0, 0};
    if (slot < 8) r.push_v = slot;
    else if (x >= 4 && slot < 24) { r.ti = x; r.tj = slot - 8; r.half = -1; }
    else { const int q = slot & 7; r.ti = q >> 1; r.half = q & 1; r.tj = x < 4 ? 3 * x + ((slot - 8) >> 3) : 8 + x; }
    return r;
}

constexpr unsigned ENT_NONE = 0xFFFFu;

__device__ __forceinline__ unsigned call_number(const Params& P) { return (unsigned)uni((int)ld_agt((const unsigned*)(P.ws[P.rank] + WS_DONE))) / (unsigned)GRID + 1u; }

__device__ __forceinline__ unsigned pusher(const Params& P, LAS unsigned char* lds, int link, int v) {
    const int tid = threadIdx.x, wid = uni(tid >> 6), lane = tid & 63;
    unsigned call = 0; int par = 0;
    const int grp = grp_of(P.rank), u = idx_of(P.rank);
    typedef const __attribute__((address_space(4))) unsigned* ctab_t;
    int dest, len; ctab_t tab;
    if (link < 3) { dest = rank_of(grp, (u + 1 + link) & 3); tab = (ctab_t)TAB_INTRA + u * TAB_INTRA_WORDS; len = TAB_INTRA_LEN; }
    else if (link == 3) { dest = P.rank; tab = (ctab_t)TAB_SELF; len = TAB_SELF_LEN; }
    else { dest = rank_of(grp ^ 1, link & 3); tab = (ctab_t)TAB_INTER + (link & 3) * TAB_INTER_WORDS; len = TAB_INTER_LEN; }
    char* const ws = P.ws[P.rank];
    char* const dws = P.ws[dest];
    unsigned* const tmo = (unsigned*)(ws + WS_TMO);
    LAS unsigned char* mybuf = lds + wid * 16384;
    bool seeds_ok = false;
    auto entry = [&](int i) -> unsigned { return i < len ? (tab[i >> 1] >> ((i & 1) << 4)) & 0xFFFFu : ENT_NONE; };
    const unsigned vsrcW = lane * 16, vsrcX = (lane >> 3) * (KS * 4) + (lane & 7) * 32;
    const unsigned vdstX = (lane >> 3) * 128 + (((lane & 7) ^ (lane >> 3)) << 4);
    auto issue_loads = [&](unsigned e, int buf) {
        const int kind = e & 3, tile = (e >> 2) & 15, s = (e >> 6) & 15;
        const char* src; unsigned voff; size_t s1, s2;
        if (kind == 1) { src = (const char*)P.w + ((size_t)(s * 64 + wid * 8) * NG + (size_t)(grp_of(dest) * 4096 + tile * 256)) * 4; voff = vsrcW; s1 = 32768; s2 = 65536; }
        else if (kind == 0) { src = (const char*)P.x + ((size_t)(row_base(dest, tile) + wid * 32) * KS + s * 64) * 4; voff = vsrcX; s1 = 16; s2 = 32768; }
        else {
            if (!seeds_ok) {
                const int q = lane & (RELAY_PAD - 1), sv = lane / RELAY_PAD;
                const unsigned se = RELAY_SET[u * RELAY_PAD + q];
                for (int pass = 0; pass < 4 * RELAY_PAD / 64; ++pass) {
                    const int srcr = rank_of(grp ^ 1, sv + pass * (64 / RELAY_PAD));
                    const unsigned* f = (const unsigned*)(ws + WS_FLW) + ((se >> 4) * NR + srcr) * 32 + (se & 15u);
                    const unsigned long long t0 = rtc();
                    while (!__all(se == 0xFFFFu || ld_sys(f) >= call)) {
                        if ((unsigned)uni((int)ld_agt(tmo)) == call || rtc() - t0 > DEADLINE) { if (lane == 0) st_agt(tmo, call); break; }
                        __builtin_amdgcn_s_sleep(2);
                    }
                }
                CFENCE; seeds_ok = true;
            }
            src = ws + WS_W + par * WBUF + (size_t)(tile * NKT + rank_of(grp ^ 1, (e >> 10) & 3) * 16 + s) * BLK + wid * 4096; voff = vsrcW; s1 = 1024; s2 = 0;
        }
        unsigned vo = voff; asm volatile("" : "+v"(vo));
#pragma unroll
        for (int i = 0; i < 8; ++i) {
            const size_t off = kind == 2 ? (size_t)(i & 3) * 1024 : (i >> 1) * s2 + (i & 1) * s1;
            __builtin_amdgcn_global_load_lds((const unsigned*)(opaque(src + off) + vo), (LAS unsigned*)(mybuf + buf * 8192 + i * 1024), 16, 0, 17);
        }
    };
    auto do_stores = [&](unsigned e, int buf) -> unsigned* {
        const int kind = e & 3, tile = (e >> 2) & 15, s = (e >> 6) & 15;
        const int srcrank = kind == 2 ? rank_of(grp ^ 1, (e >> 10) & 3) : P.rank, kt = srcrank * 16 + s;
        const LAS unsigned char* b = mybuf + buf * 8192 + lane * 16;
        if (kind == 1) {
            char* d = dws + WS_W + par * WBUF + (size_t)(tile * NKT + kt) * BLK + (lane >> 5) * HTB + wid * 2048;
            f32x4 x[8];
#pragma unroll
            for (int i = 0; i < 8; ++i) x[i] = *(const LAS f32x4*)(b + i * 1024) * W_SCALE;
#pragma unroll
            for (int j = 0; j < 4; ++j) {
                u32x4 o; o.x = pk2(x[0][j], x[1][j]); o.y = pk2(x[2][j], x[3][j]); o.z = pk2(x[4][j], x[5][j]); o.w = pk2(x[6][j], x[7][j]);
                st16_wt(d + (j * 32 + (((lane & 31) + 4 * j) & 31)) * 16, o);
            }
            return (unsigned*)(dws + WS_FLW) + (tile * NR + srcrank) * 32 + s;
        } else if (kind == 0) {
            char* d = dws + WS_X + par * XBUF + (size_t)(tile * NKT + kt) * BLK + (wid >> 2) * HTB + (wid & 3) * 4096 + vdstX;
#pragma unroll
            for (int t = 0; t < 4; ++t) {
                const f32x4 lo = *(const LAS f32x4*)(b + (2 * t) * 1024), hi = *(const LAS f32x4*)(b + (2 * t + 1) * 1024);
                u32x4 o; o.x = pk2(lo[0], lo[1]); o.y = pk2(lo[2], lo[3]); o.z = pk2(hi[0], hi[1]); o.w = pk2(hi[2], hi[3]);
                st16_wt(d + t * 1024, o);
            }
            return (unsigned*)(dws + WS_FLX) + (tile * NR + srcrank) * 32 + s;
        } else {
            char* d = dws + WS_W + par * WBUF + (size_t)(tile * NKT + kt) * BLK + wid * 4096 + lane * 16;
#pragma unroll
            for (int t = 0; t < 4; ++t) st16_wt(d + t * 1024, *(const LAS u32x4*)(b + t * 1024));
            return (unsigned*)(dws + WS_FLW) + (tile * NR + srcrank) * 32 + s;
        }
    };

    unsigned e = (((const unsigned*)P.first)[(link * 8 + v) >> 1] >> ((v & 1) << 4)) & 0xFFFFu;
    if (e != ENT_NONE && (e & 3u) != 2u) issue_loads(e, 0);
    call = call_number(P); par = call & 1;
    if (e == ENT_NONE) return call;
    if ((e & 3u) == 2u) issue_loads(e, 0);
    unsigned* flag1 = nullptr;
    for (int k = 0;; ++k) {
        const unsigned en = entry(v + (k + 1) * PG);
        if (en != ENT_NONE) { issue_loads(en, (k + 1) & 1); if (k == 0) WAIT_V(8); else WAIT_V(12); }
        else WAIT_V(0);
        unsigned* const flag0 = do_stores(e, k & 1);
        if (k >= 1) {
            if (en != ENT_NONE) WAIT_V(12); else WAIT_V(0);
            BAR; CFENCE;
            if (tid == 0) st_sys(flag1, call);
        }
        flag1 = flag0;
        if (en == ENT_NONE) break;
        e = en;
    }
    WAIT_V(0); BAR; CFENCE;
    if (tid == 0) st_sys(flag1, call);
    return call;
}

constexpr int LD_AUX = 17;
constexpr int POLL_MIN = 4;

__device__ __forceinline__ float silu(float acc) {
    const float y = acc * (1.0f / W_SCALE);
    const float c = __builtin_amdgcn_fmed3f(y, -60.0f, 60.0f);
    return y * __builtin_amdgcn_rcpf(1.0f + __expf(-c));
}

struct Stream {
    unsigned long long avLo, avHi, isLo, isHi;
    int n_issued, done; bool refresh;
};
#define PICK(k) do { if (S.avLo) { k = __builtin_ctzll(S.avLo); S.avLo &= S.avLo - 1; S.isLo |= 1ull << k; } \
        else { k = __builtin_ctzll(S.avHi); S.avHi &= S.avHi - 1; S.isHi |= 1ull << k; k += 64; } } while (0)
#define FLAGS_FETCH(area) do { if (wid == 0) { \
            __builtin_amdgcn_global_load_lds((const unsigned*)((const char*)fX + foff), (LAS unsigned*)(lds + LDS_FLG + (area) * 1024), 4, 0, 17); \
            __builtin_amdgcn_global_load_lds((const unsigned*)((const char*)fX + 512 + foff), (LAS unsigned*)(lds + LDS_FLG + (area) * 1024 + 256), 4, 0, 17); \
            __builtin_amdgcn_global_load_lds((const unsigned*)((const char*)fW + foff), (LAS unsigned*)(lds + LDS_FLG + (area) * 1024 + 512), 4, 0, 17); \
            __builtin_amdgcn_global_load_lds((const unsigned*)((const char*)fW + 512 + foff), (LAS unsigned*)(lds + LDS_FLG + (area) * 1024 + 768), 4, 0, 17); \
        } } while (0)
#define FLAGS_TAKE(area) do { \
            const LAS unsigned* fl = (const LAS unsigned*)(lds + LDS_FLG + (area) * 1024) + lane; \
            const unsigned x0 = fl[0], x1 = fl[64], w0 = fl[128], w1 = fl[192]; \
            S.avLo |= __ballot(x0 >= call && w0 >= call) & ~S.isLo; S.avHi |= __ballot(x1 >= call && w1 >= call) & ~S.isHi; } while (0)
#define NAVAIL() (__builtin_popcountll(S.avLo) + __builtin_popcountll(S.avHi))
#define STEP_TOP(rd, wr_, idle) \
        if (S.refresh) FLAGS_TAKE(rd); \
        if ((S.avLo | S.avHi) == 0ull && (idle)) { \
            if (wid == 0) { \
                const unsigned long long t0 = rtc(); unsigned long long lo, hi; \
                for (;;) { \
                    const unsigned x0 = ld_sys((const unsigned*)((const char*)fX + foff)), x1 = ld_sys((const unsigned*)((const char*)fX + 512 + foff)); \
                    const unsigned w0 = ld_sys((const unsigned*)((const char*)fW + foff)), w1 = ld_sys((const unsigned*)((const char*)fW + 512 + foff)); \
                    lo = __ballot(x0 >= call && w0 >= call) & ~S.isLo; hi = __ballot(x1 >= call && w1 >= call) & ~S.isHi; \
                    if ((lo | hi) != 0ull) break; \
                    if ((unsigned)uni((int)ld_agt(tmo)) == call || rtc() - t0 > DEADLINE) { if (lane == 0) st_agt(tmo, call); lo = ~S.isLo; hi = ~S.isHi; break; } \
                    __builtin_amdgcn_s_sleep(2); \
                } \
                if (lane == 0) *mslot = (u32x4){(unsigned)lo, (unsigned)(lo >> 32), (unsigned)hi, (unsigned)(hi >> 32)}; \
                asm volatile("s_waitcnt vmcnt(0) lgkmcnt(0)" ::: "memory"); \
            } \
            CFENCE; BAR; CFENCE; \
            const u32x4 mv = *mslot; \
            S.avLo = ((unsigned long long)(unsigned)uni((int)mv.y) << 32) | (unsigned)uni((int)mv.x); \
            S.avHi = ((unsigned long long)(unsigned)uni((int)mv.w) << 32) | (unsigned)uni((int)mv.z); \
            WAIT_L(0); \
        } \
        bool vn = true; int kn = 0; \
        if (S.avLo | S.avHi) { PICK(kn); } else vn = false; \
        S.n_issued += vn; \
        { const int navail = NAVAIL(); S.refresh = navail < POLL_MIN && S.n_issued + navail < NKT; } \
        if (S.refresh) FLAGS_FETCH(wr_);
#define GEMM_COMMON \
    const int tid = threadIdx.x, wid = uni(tid >> 6), lane = tid & 63, wr = wid >> 2, wc = wid & 3, fr = lane & 15, fq = lane >> 4; \
    char* ws = P.ws[P.rank]; \
    const char* wb = ws + WS_W + (call & 1) * WBUF + (size_t)tj * NKT * BLK; \
    const unsigned* fX = (const unsigned*)(ws + WS_FLX) + ti * NR * 32; \
    const unsigned* fW = (const unsigned*)(ws + WS_FLW) + tj * NR * 32; \
    unsigned* tmo = (unsigned*)(ws + WS_TMO); \
    LAS u32x4* mslot = (LAS u32x4*)(lds + LDS_CTL); \
    const unsigned voff = (unsigned)tid * 16u; \
    const unsigned ldsw = (unsigned)wid * 1024u; \
    const unsigned foff = ((lane >> 4) * 32 + (lane & 15)) * 4;            \
    const int aoff0 = (wr * 64 + fr) * 128 + ((fq ^ (fr & 7)) << 4), aoff1 = aoff0 ^ 64; \
    const int boff0 = fq * 2048 + ((fr & 3) * 32 + ((wc * 8 + (fr >> 2) + 4 * (fr & 3)) & 31)) * 16; \
    const int boff1 = fq * 2048 + ((fr & 3) * 32 + ((wc * 8 + 4 + (fr >> 2) + 4 * (fr & 3)) & 31)) * 16; \
    h16x8 At[4][2], B0[2][2], B1[2][2]; \
    Stream S = {0ull, 0ull, 0ull, 0ull, 0, 0, false}
#define STAGE(q, gbase) do { unsigned _v = voff; asm volatile("" : "+v"(_v)); _Pragma("unroll") for (int _i = 0; _i < 2; ++_i) \
        __builtin_amdgcn_global_load_lds((const unsigned*)(opaque((gbase) + _i * 8192) + _v), (LAS unsigned*)(lds + (q) * HTB + ldsw + _i * 8192), 16, 0, LD_AUX); } while (0)
#define LDA(dst, q) do { _Pragma("unroll") for (int m = 0; m < 4; ++m) { dst[m][0] = *(const LAS h16x8*)(lds + (q) * HTB + aoff0 + m * 2048); dst[m][1] = *(const LAS h16x8*)(lds + (q) * HTB + aoff1 + m * 2048); } } while (0)
#define LDB(dst, q) do { _Pragma("unroll") for (int k = 0; k < 2; ++k) { dst[0][k] = *(const LAS h16x8*)(lds + (q) * HTB + boff0 + k * 8192); dst[1][k] = *(const LAS h16x8*)(lds + (q) * HTB + boff1 + k * 8192); } } while (0)
#define MMA(C, At, Bt) do { __builtin_amdgcn_s_setprio(1); _Pragma("unroll") for (int m = 0; m < 4; ++m) _Pragma("unroll") for (int n = 0; n < 2; ++n) _Pragma("unroll") for (int k = 0; k < 2; ++k) \
        C[m][n] = __builtin_amdgcn_mfma_f32_16x16x32_f16(Bt[n][k], At[m][k], C[m][n], 0, 0, 0); __builtin_amdgcn_s_setprio(0); } while (0)
#define MMA_H(C, At, Bt, m0) do { _Pragma("unroll") for (int m = m0; m < m0 + 2; ++m) _Pragma("unroll") for (int n = 0; n < 2; ++n) _Pragma("unroll") for (int k = 0; k < 2; ++k) \
        C[m][n] = __builtin_amdgcn_mfma_f32_16x16x32_f16(Bt[n][k], At[m][k], C[m][n], 0, 0, 0); } while (0)
#define MMA_S(C, At, Bt, STG) do { __builtin_amdgcn_s_setprio(1); MMA_H(C, At, Bt, 0); SCHED; STG; SCHED; MMA_H(C, At, Bt, 2); __builtin_amdgcn_s_setprio(0); } while (0)
#define ZERO4(C) do { _Pragma("unroll") for (int m = 0; m < 4; ++m) _Pragma("unroll") for (int n = 0; n < 2; ++n) C[m][n] = (f32x4){0.f, 0.f, 0.f, 0.f}; } while (0)
#define SLAB_OUT(C0, C1, rowptr, STORE) do { \
        _Pragma("unroll") for (int m = 0; m < 4; ++m) { \
            const int r = wr * 64 + m * 16 + fr; \
            _Pragma("unroll") for (int bj = 0; bj < 2; ++bj) _Pragma("unroll") for (int n = 0; n < 2; ++n) { \
                f32x4 y = bj ? C1[m][n] : C0[m][n]; \
                y.x = silu(y.x); y.y = silu(y.y); y.z = silu(y.z); y.w = silu(y.w); \
                if (bad) y = (f32x4){qnan, qnan, qnan, qnan}; \
                const int ch = bj * 32 + wc * 8 + n * 4 + fq; \
                *(LAS f32x4*)(lds + r * 1024 + ((ch ^ (r & 7)) << 4)) = y; \
            } } \
        WAIT_L(0); BAR; CFENCE; \
        _Pragma("unroll") for (int q = 0; q < 16; ++q) { \
            const int r = wid * 16 + q; \
            const u32x4 y = *(const LAS u32x4*)(lds + r * 1024 + ((lane ^ (r & 7)) << 4)); \
            STORE((rowptr) + (size_t)r * (NG * 4) + lane * 16, y); \
        } } while (0)

__device__ __forceinline__ void gemm_sent(const Params& P, LAS unsigned char* lds, int ti, int tj, unsigned call) {
    GEMM_COMMON;
    const char* xb = ws + WS_X + (call & 1) * XBUF + (size_t)ti * NKT * BLK;
    f32x4 acc[2][2][4][2];
    ZERO4(acc[0][0]); ZERO4(acc[0][1]); ZERO4(acc[1][0]); ZERO4(acc[1][1]);
    bool v0 = false, v1 = false;
    const char* apend = xb;
    if (wr == 1) BAR;
#define SA(b, h) (2 * (b) + (h))
#define SB(b, h) (4 + 2 * (b) + (h))
#define HALF(b, o, vcur, voth) do { \
        STEP_TOP(o, b, !(vcur) && !(voth)) \
        const char* an = xb + (size_t)kn * BLK; const char* bn = wb + (size_t)kn * BLK; \
        const int n4 = S.refresh ? ((voth) ? 2 : 0) + (vn ? 6 : 0) : (vn ? 10 : 4); \
        if (vcur) { \
            LDB(B0, SB(b, 0)); SCHED; LDA(At, SA(b, 0)); if (voth) { STAGE(SA(o, 1), apend); WAIT_V(10); } else { WAIT_V(2); } \
            WAIT_L(8); BAR; WAIT_L(0); MMA(acc[0][0], At, B0); BAR; SCHED; \
            LDB(B1, SB(b, 1)); if (vn) STAGE(SB(b, 0), bn); \
            WAITN(((voth) ? 8 : 0) + (vn ? 2 : 0)); \
            BAR; WAIT_L(0); MMA(acc[0][1], At, B1); BAR; \
            LDA(At, SA(b, 1)); if (vn) STAGE(SA(b, 0), an); \
            BAR; WAIT_L(0); MMA(acc[1][0], At, B0); BAR; SCHED; \
            if (vn) STAGE(SB(b, 1), bn + HTB); \
            WAITN(n4); \
            BAR; MMA(acc[1][1], At, B1); BAR; \
            ++S.done; \
        } else { \
            if (voth) STAGE(SA(o, 1), apend); \
            BAR; BAR; \
            if (vn) STAGE(SB(b, 0), bn); \
            BAR; BAR; \
            if (vn) STAGE(SA(b, 0), an); \
            BAR; BAR; \
            if (vn) STAGE(SB(b, 1), bn + HTB); \
            WAITN(n4); \
            BAR; BAR; \
        } \
        apend = an + HTB; vcur = vn; } while (0)
    for (;;) {
        int navail = NAVAIL();
        if (v0 && v1 && !S.refresh && navail >= 2) {
            int k2, k3; PICK(k2); PICK(k3); navail -= 2;
            for (;;) {
                const bool poll = navail < POLL_MIN && S.n_issued + 2 + navail < NKT;
                const char* a2 = xb + (size_t)k2 * BLK; const char* b2 = wb + (size_t)k2 * BLK;
                const char* a3 = xb + (size_t)k3 * BLK; const char* b3 = wb + (size_t)k3 * BLK;
                LDB(B0, SB(0, 0)); SCHED; LDA(At, SA(0, 0)); STAGE(SA(1, 1), apend); WAIT_V(10);
                WAIT_L(8); BAR; WAIT_L(0); MMA(acc[0][0], At, B0); BAR; SCHED;
                LDB(B1, SB(0, 1)); STAGE(SB(0, 0), b2); WAIT_V(10);
                BAR; WAIT_L(0); MMA(acc[0][1], At, B1); BAR;
                LDA(At, SA(0, 1)); STAGE(SA(0, 0), a2);
                BAR; WAIT_L(0); MMA(acc[1][0], At, B0); BAR; SCHED;
                STAGE(SB(0, 1), b2 + HTB); WAIT_V(10);
                BAR; MMA(acc[1][1], At, B1); BAR;
                if (poll) FLAGS_FETCH(3);
                LDB(B0, SB(1, 0)); SCHED; LDA(At, SA(1, 0)); STAGE(SA(0, 1), a2 + HTB); WAIT_V(10);
                WAIT_L(8); BAR; WAIT_L(0); MMA(acc[0][0], At, B0); BAR; SCHED;
                LDB(B1, SB(1, 1)); STAGE(SB(1, 0), b3); WAIT_V(10);
                BAR; WAIT_L(0); MMA(acc[0][1], At, B1); BAR;
                LDA(At, SA(1, 1)); STAGE(SA(1, 0), a3); if (poll) WAIT_V(6);
                BAR; WAIT_L(0); MMA(acc[1][0], At, B0); BAR; SCHED;
                STAGE(SB(1, 1), b3 + HTB);
                apend = a3 + HTB; S.done += 2; S.n_issued += 2;
                if (poll) { FLAGS_TAKE(3); navail = NAVAIL(); }
                const bool more = navail >= 2;
                if (more) { PICK(k2); PICK(k3); navail -= 2; }
                WAIT_V(10);
                BAR; MMA(acc[1][1], At, B1); BAR;
                if (!more) break;
            }
        }
        HALF(0, 1, v0, v1);
        if (S.done >= NKT) break;
        HALF(1, 0, v1, v0);
        if (S.done >= NKT) break;
    }
#undef HALF
#undef SA
#undef SB
    WAIT_V(0);
    if (wr == 0) BAR;
    BAR;
    const bool bad = (unsigned)uni((int)ld_agt(tmo)) == call;
    const int grow = row_base(P.rank, ti), owner = grow >> 10;
    char* obase = (char*)P.out[owner] + ((size_t)(grow & 1023) * NG + (size_t)(grp_of(P.rank) * 4096 + tj * 256)) * 4;
    const float qnan = __builtin_nanf("");
    SLAB_OUT(acc[0][0], acc[0][1], obase, st16_wt);
    WAIT_L(0); CFENCE; BAR; CFENCE;
    SLAB_OUT(acc[1][0], acc[1][1], obase + (size_t)128 * (NG * 4), st16_wt);
    WAIT_V(0); BAR; CFENCE;
    if (tid < 2) st_sys((unsigned*)(P.ws[owner] + WS_FLO) + idx_of(P.rank) * 32 + tj * 2 + tid, call);
}

__device__ __forceinline__ void gemm_half(const Params& P, LAS unsigned char* lds, int ti, int half, int tj, unsigned call) {
    GEMM_COMMON;
    const char* xb = ws + WS_X + (call & 1) * XBUF + (size_t)ti * NKT * BLK + half * HTB;
    f32x4 acc[2][4][2];
    ZERO4(acc[0]); ZERO4(acc[1]);
    bool v0 = false, v1 = false, v2 = false;
    if (wr == 1) BAR;
#define HA(b) (3 * (b))
#define HB(b, h) (3 * (b) + 1 + (h))
#define THIRD(b, nb, vcur, vnx, vnew) do { \
        STEP_TOP(nb, b, !(vcur) && !(vnx)) \
        const char* an = xb + (size_t)kn * BLK; const char* bn = wb + (size_t)kn * BLK; \
        const int n1 = ((vnx) ? 6 : 0) + (vn ? 4 : 0), n2 = (S.refresh || !(vnx) ? 0 : 2) + (vn ? 6 : 0); \
        if (vcur) { \
            LDB(B0, HB(b, 0)); LDA(At, HA(b)); if (vn) { STAGE(HA(nb), an); STAGE(HB(nb, 0), bn); } \
            WAITN(n1); \
            BAR; WAIT_L(0); MMA(acc[0], At, B0); BAR; SCHED; \
            LDB(B1, HB(b, 1)); if (vn) STAGE(HB(nb, 1), bn + HTB); \
            WAITN(n2); \
            BAR; WAIT_L(0); MMA(acc[1], At, B1); BAR; SCHED; \
            ++S.done; \
        } else { \
            if (vn) { STAGE(HA(nb), an); STAGE(HB(nb, 0), bn); } \
            BAR; BAR; \
            if (vn) STAGE(HB(nb, 1), bn + HTB); \
            WAITN(n2); \
            BAR; BAR; \
        } \
        vnew = vn; } while (0)
#define FAST3(b, nb, k, MID) do { \
        const char* an = xb + (size_t)(k) * BLK; const char* bn = wb + (size_t)(k) * BLK; \
        LDB(B0, HB(b, 0)); LDA(At, HA(b)); WAIT_V(6); \
        BAR; WAIT_L(0); MMA_S(acc[0], At, B0, { STAGE(HA(nb), an); STAGE(HB(nb, 0), bn); }); BAR; SCHED; \
        LDB(B1, HB(b, 1)); MID; WAIT_V(6); \
        BAR; WAIT_L(0); MMA_S(acc[1], At, B1, STAGE(HB(nb, 1), bn + HTB)); BAR; SCHED; } while (0)
    for (;;) {
        int navail = NAVAIL();
        if (v0 && v1 && !S.refresh && navail >= 3) {
            int k2, k3, k4; PICK(k2); PICK(k3); PICK(k4); navail -= 3;
            for (;;) {
                const bool poll = navail < POLL_MIN + 1 && S.n_issued + 3 + navail < NKT;
                bool more; const int c2 = k2, c3 = k3, c4 = k4;
                FAST3(0, 2, c2, );
                if (poll) FLAGS_FETCH(3);
                FAST3(1, 0, c3, );
                FAST3(2, 1, c4, { S.done += 3; S.n_issued += 3; if (poll) { FLAGS_TAKE(3); navail = NAVAIL(); } more = navail >= 3; if (more) { PICK(k2); PICK(k3); PICK(k4); navail -= 3; } });
                if (!more) break;
            }
        }
        THIRD(0, 2, v0, v1, v2);
        if (S.done >= NKT) break;
        THIRD(1, 0, v1, v2, v0);
        if (S.done >= NKT) break;
        THIRD(2, 1, v2, v0, v1);
        if (S.done >= NKT) break;
    }
#undef THIRD
#undef FAST3
#undef HA
#undef HB
    WAIT_V(0);
    if (wr == 0) BAR;
    BAR;
    const bool bad = (unsigned)uni((int)ld_agt(tmo)) == call;
    const int grow = row_base(P.rank, ti) + half * 128, owner = grow >> 10;
    char* obase = (char*)P.out[owner] + ((size_t)(grow & 1023) * NG + (size_t)(grp_of(P.rank) * 4096 + tj * 256)) * 4;
    const float qnan = __builtin_nanf("");
    SLAB_OUT(acc[0], acc[1], obase, st16_wt);
    if (owner != P.rank) {
        WAIT_V(0); BAR; CFENCE;
        if (tid == 0) st_sys((unsigned*)(P.ws[owner] + WS_FLO) + idx_of(P.rank) * 32 + tj * 2 + half, call);
    }
}

__global__ void __launch_bounds__(512, 2) fused(Params P) {
    extern __shared__ __attribute__((aligned(16))) unsigned char shm[];
    LAS unsigned char* lds = (LAS unsigned char*)shm;
    char* ws = P.ws[P.rank];
    const int b = blockIdx.x, xcd = (int)(__builtin_amdgcn_s_getreg(20 | (3 << 11)) & 7u);
    const Role role = role_of(xcd, b >> 3);
    unsigned call;
    if (role.push_v >= 0) {
        call = pusher(P, lds, xcd, role.push_v);
        __syncthreads();
    } else call = call_number(P);
    if (role.ti >= 0) { if (role.half < 0) gemm_sent(P, lds, role.ti, role.tj, call); else gemm_half(P, lds, role.ti, role.half, role.tj, call); }
    if (xcd == 4 && b < 8) {
        if (threadIdx.x < 64) {
            const int lane = threadIdx.x;
            const unsigned* f = (const unsigned*)(ws + WS_FLO) + lane;
            unsigned* tmo = (unsigned*)(ws + WS_TMO);
            const unsigned long long t0 = rtc();
            for (;;) {
                if (__all(ld_sys(f) >= call && ld_sys(f + 64) >= call)) break;
                if ((unsigned)uni((int)ld_agt(tmo)) == call || rtc() - t0 > DEADLINE) { if (lane == 0) st_agt(tmo, call); break; }
                __builtin_amdgcn_s_sleep(2);
            }
            if ((unsigned)uni((int)ld_agt(tmo)) == call && lane == 0)
                P.out[P.rank][(size_t)grp_of(P.rank) * 4096] = __builtin_nanf("");
        }
    }
    __syncthreads();
    if (threadIdx.x == 0) __hip_atomic_fetch_add((gu32*)(ws + WS_DONE), 1u, __ATOMIC_RELAXED, __HIP_MEMORY_SCOPE_AGENT);
}

}

extern "C" void kernel_launch(int rank, int nranks, ncclComm_t comm, hipStream_t stream,
                              void* const* d_in, const int* in_sizes, int n_in,
                              void* d_out, int out_size,
                              const int* in_dtypes, int out_dtype,
                              void* const* d_out_all,
                              void* const* d_ws_all, size_t ws_bytes) {
    using namespace gk;
    (void)comm; (void)in_sizes; (void)n_in; (void)d_out; (void)out_size; (void)in_dtypes; (void)out_dtype;
    if (nranks != NR || ws_bytes < WS_END || rank < 0 || rank >= NR) { fprintf(stderr, "unsupported configuration\n"); return; }
    static bool attr_set[64] = {};
    int dev = 0; HIP_CHECK(hipGetDevice(&dev));
    if (dev >= 0 && dev < 64 && !attr_set[dev]) {
        HIP_CHECK(hipFuncSetAttribute((const void*)fused, hipFuncAttributeMaxDynamicSharedMemorySize, LDS_BYTES));
        attr_set[dev] = true;
    }
    Params P;
    P.x = (const float*)d_in[0]; P.w = (const float*)d_in[1];
    for (int p = 0; p < NR; ++p) { P.ws[p] = (char*)d_ws_all[p]; P.out[p] = (float*)d_out_all[p]; }
    P.rank = rank; P.pad = 0;
    for (int i = 0; i < 64; ++i) P.first[i] = FIRST_ENT[idx_of(rank)][i];
    hipLaunchKernelGGL(fused, dim3(GRID), dim3(512), LDS_BYTES, stream, P);
    HIP_CHECK(hipGetLastError());
}
```
